```python
import numpy as np

kernel_name = "gpu_ag_gemm_m2048_k2048_n2048_f32_silu"


ABSMAX_RTOL = 0.001

COMPUTE_DTYPE = "float32"
EPILOGUE = "silu"
IN_QUANT = None
OUT_QUANT = None
IN_DTYPES = {"x": "float32", "w_mat": "float32"}
WORLD_SIZE = 8

def setup_inputs():

    x = np.random.randn(2048, 2048).astype(np.float32)
    w_mat = (np.random.randn(2048, 2048) / np.sqrt(2048)).astype(np.float32)
    return {"x": x, "w_mat": w_mat}

def _epilogue(y):
    return (y / (1.0 + np.exp(-np.clip(y, -60.0, 60.0)))).astype(np.float32)

def reference(x, w_mat):
    y = (x @ w_mat).astype(np.float32)
    return _epilogue(y)
```

```cpp
#include <hip/hip_runtime.h>
#include <rccl/rccl.h>
#include <cstdio>

#define HIP_CHECK(expr) do { hipError_t _e = (expr); if (_e != hipSuccess) { \
    fprintf(stderr, "HIP error %s at %s:%d\n", hipGetErrorString(_e), __FILE__, __LINE__); \
    return; } } while (0)

#define GAS __attribute__((address_space(1)))
typedef _Float16 half8 __attribute__((ext_vector_type(8)));
typedef _Float16 half2v __attribute__((ext_vector_type(2)));
typedef float f32x16 __attribute__((ext_vector_type(16)));
typedef float f32x4 __attribute__((ext_vector_type(4)));
typedef float f32x2 __attribute__((ext_vector_type(2)));
typedef unsigned u32x4 __attribute__((ext_vector_type(4)));

#define SEND_AUX 17
#define POLL_AUX 17
#define WINDOW 4
#define NSW 2
#define LOOKAHEAD 2
#define POLL_SLEEP 4
#define DESTMAP 0

constexpr int NR = 8, MSH = 256, KD = 2048, NSH = 256;
constexpr int NCW = 4;
constexpr int NWAVES = NCW + NSW;
constexpr int NTHREADS = NWAVES * 64;
constexpr int NWG = 256;
constexpr unsigned FRAG = 1024;
constexpr unsigned SRC_BYTES = 1u << 20;
constexpr unsigned SLOT_BYTES = NR * SRC_BYTES;
constexpr int NSLOT = 3;
constexpr unsigned CTR_OFF = NSLOT * SLOT_BYTES;
constexpr unsigned TMO_OFF = CTR_OFF + NWG * 128;
constexpr unsigned STAMP_OFF = TMO_OFF + 4096;
constexpr unsigned WS_NEED = STAMP_OFF + NWG * 64 * 8 * 8;
constexpr unsigned long long DEADLINE_TICKS = 150000000ull;
constexpr int ROUNDS = 32 / NSW;
constexpr int LDS_BYTES = 128 * 1024 + 64;

struct Params {
    const float* x; const float* w; float* out;
    char* ws[NR];
    int rank;
};

__device__ __forceinline__ unsigned pack2(float a, float b) {
    half2v h = __builtin_convertvector((f32x2){a, b}, half2v);
    return __builtin_bit_cast(unsigned, h);
}
__device__ __forceinline__ unsigned nz(unsigned v) { return v ? v : 0x8000u; }

#define STAMP(idx) do {} while (0)

__global__ void __launch_bounds__(NTHREADS) ag_gemm_silu(Params p) {
    extern __shared__ __attribute__((aligned(16))) char lds[];
    const int b = blockIdx.x;
    const int wave = __builtin_amdgcn_readfirstlane(threadIdx.x >> 6);
    const int lane = threadIdx.x & 63;
    char* const myws = p.ws[p.rank];
    unsigned* ctr = (unsigned*)(myws + CTR_OFF + b * 128) + wave;
    const unsigned long long t0 = __builtin_amdgcn_s_memrealtime();
    STAMP(0);
    if (wave >= NCW) {
        const int sw = wave - NCW;
        const int dsel = b & 7, g = b >> 3;
        const unsigned dest = (unsigned)(p.rank + dsel) & 7u;
        const int rt = g & 7, kq = g >> 3;
        const float* src = p.x + (size_t)(rt * 32 + (lane & 31)) * KD + kq * 16 + (lane >> 5) * 8;
        const unsigned c0 = __hip_atomic_load((GAS unsigned*)ctr, __ATOMIC_RELAXED, __HIP_MEMORY_SCOPE_AGENT);
        f32x4 raw[ROUNDS][2];
#pragma unroll
        for (int i = 0; i < ROUNDS; ++i) {
            const int r = i * NSW + sw;
            raw[i][0] = *(const f32x4*)(src + r * 64);
            raw[i][1] = *(const f32x4*)(src + r * 64 + 4);
        }
        __builtin_amdgcn_s_barrier();
        const unsigned call = __builtin_amdgcn_readfirstlane(c0) + 1;
        const unsigned slot = call % NSLOT;
        char* dbase = p.ws[dest] + slot * SLOT_BYTES + p.rank * SRC_BYTES;
        auto rs = __builtin_amdgcn_make_buffer_rsrc(dbase, 0, SRC_BYTES, 0x00020000);
#pragma unroll
        for (int i = 0; i < ROUNDS; ++i) {
            const int r = i * NSW + sw;
            u32x4 v = {nz(pack2(raw[i][0].x, raw[i][0].y)), nz(pack2(raw[i][0].z, raw[i][0].w)),
                       nz(pack2(raw[i][1].x, raw[i][1].y)), nz(pack2(raw[i][1].z, raw[i][1].w))};
            if (i >= WINDOW) asm volatile("s_waitcnt vmcnt(%0)" :: "n"(WINDOW - 1) : "memory");
            __builtin_amdgcn_raw_buffer_store_b128(v, rs, ((4 * r + kq) * 8 + rt) * FRAG + lane * 16, 0, SEND_AUX);
            if (i == 0) STAMP(1);
        }
        STAMP(2);
        if (lane == 0) *ctr = call;
        return;
    }
    const int srcsel = b & 7, j = b >> 3;
    const int src = (p.rank + srcsel) & 7;
    const int rt2 = j & 3, ct = j >> 2;
    if (threadIdx.x == 0) *(unsigned*)(lds + 128 * 1024) = 0;
    asm volatile("s_waitcnt lgkmcnt(0)" ::: "memory");
    __builtin_amdgcn_s_barrier();
    const unsigned c0 = __hip_atomic_load((GAS unsigned*)ctr, __ATOMIC_RELAXED, __HIP_MEMORY_SCOPE_AGENT);
    char* myB = lds + wave * 32 * 1024;
    {
        const float* wp = p.w + (size_t)(wave * 16 + (lane >> 5) * 8) * NSH + ct * 32 + (lane & 31);
#pragma unroll 4
        for (int i = 0; i < 32; ++i) {
            float f[8];
#pragma unroll
            for (int e = 0; e < 8; ++e) f[e] = wp[(size_t)(i * 64 + e) * NSH];
            u32x4 v = {pack2(f[0], f[1]), pack2(f[2], f[3]), pack2(f[4], f[5]), pack2(f[6], f[7])};
            *(u32x4*)(myB + i * 1024 + lane * 16) = v;
        }
    }
    const unsigned call = __builtin_amdgcn_readfirstlane(c0) + 1;
    STAMP(1);
    {
        const unsigned zslot = (call + NSLOT - 1) % NSLOT;
        auto rz = __builtin_amdgcn_make_buffer_rsrc(myws + zslot * SLOT_BYTES, 0, SLOT_BYTES, 0x00020000);
        const u32x4 z = {0, 0, 0, 0};
#pragma unroll
        for (int i = 0; i < 8; ++i)
            __builtin_amdgcn_raw_buffer_store_b128(z, rz, ((b * NCW + wave) * 8 + i) * 1024 + lane * 16, 0, 17);
    }
    if (lane == 0) *ctr = call;
    const unsigned slot = call % NSLOT;
    auto ra = __builtin_amdgcn_make_buffer_rsrc(myws + slot * SLOT_BYTES + src * SRC_BYTES, 0, SRC_BYTES, 0x00020000);
    const unsigned abase = (wave * 8 + rt2 * 2) * FRAG + lane * 16;
    f32x16 acc0 = {}, acc1 = {};
    unsigned pending = 0xFFFFFFFFu;
    bool failed = false;
    while (pending) {
        asm volatile("" ::: "memory");
        int idx[LOOKAHEAD]; bool val[LOOKAHEAD];
        unsigned m = pending;
#pragma unroll
        for (int q = 0; q < LOOKAHEAD; ++q) {
            val[q] = m != 0;
            idx[q] = val[q] ? __builtin_ctz(m) : 0;
            m &= m - 1;
        }
        u32x4 a0[LOOKAHEAD], a1[LOOKAHEAD];
#pragma unroll
        for (int q = 0; q < LOOKAHEAD; ++q) {
            if (val[q]) {
                a0[q] = __builtin_amdgcn_raw_buffer_load_b128(ra, abase, idx[q] * 32 * FRAG, POLL_AUX);
                a1[q] = __builtin_amdgcn_raw_buffer_load_b128(ra, abase + FRAG, idx[q] * 32 * FRAG, POLL_AUX);
            }
        }
        bool prog = false;
#pragma unroll
        for (int q = 0; q < LOOKAHEAD; ++q) {
            if (val[q]) {
                unsigned mn = min(min(min(a0[q].x, a0[q].y), min(a0[q].z, a0[q].w)),
                                  min(min(a1[q].x, a1[q].y), min(a1[q].z, a1[q].w)));
                if (__builtin_amdgcn_ballot_w64(mn == 0) == 0) {
                    half8 bf = *(const half8*)(myB + idx[q] * 1024 + lane * 16);
                    acc0 = __builtin_amdgcn_mfma_f32_32x32x16_f16(bf, __builtin_bit_cast(half8, a0[q]), acc0, 0, 0, 0);
                    acc1 = __builtin_amdgcn_mfma_f32_32x32x16_f16(bf, __builtin_bit_cast(half8, a1[q]), acc1, 0, 0, 0);
                    pending &= ~(1u << idx[q]);
                    prog = true;
                }
            }
        }
        if (!prog) {
            if (__builtin_amdgcn_s_memrealtime() - t0 > DEADLINE_TICKS) {
                failed = true;
                if (lane == 0) __hip_atomic_store((unsigned*)(myws + TMO_OFF), call, __ATOMIC_RELAXED, __HIP_MEMORY_SCOPE_AGENT);
                break;
            }
            __builtin_amdgcn_s_sleep(POLL_SLEEP);
        }
    }
    STAMP(2);
    if (failed) {
#pragma unroll
        for (int r = 0; r < 16; ++r) { acc0[r] = __builtin_nanf(""); acc1[r] = __builtin_nanf(""); }
    }
    {
        const int m = lane & 31, h = lane >> 5;
#pragma unroll
        for (int q = 0; q < 4; ++q) {
            const int gsw = ((2 * q + h) ^ (m & 7)) * 16;
            *(f32x4*)(myB + m * 128 + gsw) = (f32x4){acc0[4 * q], acc0[4 * q + 1], acc0[4 * q + 2], acc0[4 * q + 3]};
            *(f32x4*)(myB + (32 + m) * 128 + gsw) = (f32x4){acc1[4 * q], acc1[4 * q + 1], acc1[4 * q + 2], acc1[4 * q + 3]};
        }
        asm volatile("s_waitcnt lgkmcnt(0)" ::: "memory");
        unsigned* lc = (unsigned*)(lds + 128 * 1024);
        if (lane == 0) __hip_atomic_fetch_add(lc, 1u, __ATOMIC_RELAXED, __HIP_MEMORY_SCOPE_WORKGROUP);
        while (__hip_atomic_load(lc, __ATOMIC_RELAXED, __HIP_MEMORY_SCOPE_WORKGROUP) < (unsigned)NCW) {}
        asm volatile("" ::: "memory");
#pragma unroll
        for (int it = 0; it < 2; ++it) {
            const int row = wave * 16 + it * 8 + (lane >> 3), c = lane & 7;
            const int off = row * 128 + ((c ^ (row & 7)) * 16);
            f32x4 s = *(const f32x4*)(lds + off);
#pragma unroll
            for (int ww = 1; ww < NCW; ++ww) { f32x4 t = *(const f32x4*)(lds + ww * 32 * 1024 + off); s += t; }
            f32x4 o;
#pragma unroll
            for (int e = 0; e < 4; ++e) {
                float y = s[e];
                float t = fminf(fmaxf(y, -60.f), 60.f);
                o[e] = y * __builtin_amdgcn_rcpf(1.f + __expf(-t));
            }
            *(f32x4*)(p.out + (size_t)(src * MSH + rt2 * 64 + row) * NSH + ct * 32 + c * 4) = o;
        }
    }
    STAMP(3);
}

extern "C" void kernel_launch(int rank, int nranks, ncclComm_t comm, hipStream_t stream,
                              void* const* d_in, const int* in_sizes, int n_in,
                              void* d_out, int out_size, const int* in_dtypes, int out_dtype,
                              void* const* d_out_all, void* const* d_ws_all, size_t ws_bytes) {
    if (nranks != NR || ws_bytes < WS_NEED || (unsigned)rank >= (unsigned)NR) { fprintf(stderr, "bad config\n"); return; }
    HIP_CHECK(hipFuncSetAttribute((const void*)ag_gemm_silu, hipFuncAttributeMaxDynamicSharedMemorySize, LDS_BYTES));
    Params p;
    p.x = (const float*)d_in[0]; p.w = (const float*)d_in[1]; p.out = (float*)d_out;
    for (int i = 0; i < NR; ++i) p.ws[i] = (char*)d_ws_all[i];
    p.rank = rank;
    hipLaunchKernelGGL(ag_gemm_silu, dim3(NWG), dim3(NTHREADS), LDS_BYTES, stream, p);
    HIP_CHECK(hipGetLastError());
}
```

```cpp
#include <hip/hip_runtime.h>
#include <rccl/rccl.h>
#include <cstdio>
#include <vector>
#include <algorithm>

#define HIP_CHECK(expr) do { hipError_t _e = (expr); if (_e != hipSuccess) { \
    fprintf(stderr, "HIP error %s at %s:%d\n", hipGetErrorString(_e), __FILE__, __LINE__); \
    return; } } while (0)

#define GAS __attribute__((address_space(1)))
#define LAS __attribute__((address_space(3)))
typedef _Float16 half8 __attribute__((ext_vector_type(8)));
typedef _Float16 half2v __attribute__((ext_vector_type(2)));
typedef float f32x4 __attribute__((ext_vector_type(4)));
typedef float f32x2 __attribute__((ext_vector_type(2)));
typedef unsigned u32x4 __attribute__((ext_vector_type(4)));
typedef unsigned u32x2 __attribute__((ext_vector_type(2)));
typedef unsigned long long u64;

#define NPHI 11
#define NPSI 4
#define OWNWG 3
#define WINDOW 2
#define PREF 2
#define B_SLEEP 4
#define B_CANARY 1
#define ROT 9
#define WFREE 1
#define WTOKEN 1
#define BCAN_I 7
#define XGATE 1
#define NZWG 64
#define SENDMAP 0
#define ZDELAY 0
#define ZMODE 1
#define WSTAG 3
#define XACK 7
#define WINDOW_OPP 2
#define WGRP 4
#define POLL_SLEEP 4
#define OPP_SCALE 1.3
constexpr int NRHO = 16 - NPHI - NPSI;
constexpr bool COOP = (NPHI + NPSI) > 0;
constexpr int AUX_SYS = 17;

constexpr int NR = 8, MSH = 256, KD = 2048, NSH = 256;
constexpr int NWAVES = 8, NTHREADS = 512, NWG = 256, NSENDWG = 64, NCONS = NWG - NSENDWG;
constexpr int MAXT = 4;
constexpr unsigned FRAG = 1024, RT_BYTES = 64 * FRAG;
constexpr unsigned SRC_BYTES = 1u << 20;
constexpr unsigned W_OFF = NR * SRC_BYTES;
constexpr unsigned WQ_BYTES = 256 * FRAG;
constexpr unsigned SLOT_BYTES = W_OFF + 8 * WQ_BYTES;
constexpr int NSLOT = 3;
constexpr unsigned CTR_OFF = NSLOT * SLOT_BYTES;
constexpr unsigned TMO_OFF = CTR_OFF + NWG * 128;
constexpr unsigned FLAG_OFF = TMO_OFF + 4096;
constexpr unsigned STAMP_OFF = FLAG_OFF + 4096;
constexpr unsigned WS_NEED = STAMP_OFF + NWG * 64 * 8 * 8;
constexpr u64 DEADLINE_TICKS = 150000000ull;
constexpr int LDS_BYTES = MAXT * 32 * 1024 + 64;
constexpr int NOWN = 8 * 4 * OWNWG;
constexpr int NZ = 4 * (NPHI + NRHO) + 4 * (NPSI + NRHO) + (COOP ? 32 : 16);
constexpr int NFLAGS = COOP ? 4 * 8 * (6 - OWNWG) : 0;

struct Params {
    const float* x; const float* w;
    char* myws;
    char* peer[NR];
    float* outq[8];
    unsigned* flagq[4];
    int rank; int pad;
    u64 task[NCONS];
};

__device__ __forceinline__ unsigned pack2(float a, float b) {
    half2v h = __builtin_convertvector((f32x2){a, b}, half2v);
    return __builtin_bit_cast(unsigned, h);
}
__device__ __forceinline__ unsigned nz(unsigned v) { return v ? v : 0x8000u; }
__device__ __forceinline__ bool full(const u32x4& a) {
    unsigned mn = min(min(a.x, a.y), min(a.z, a.w));
    return __builtin_amdgcn_ballot_w64(mn == 0) == 0;
}
__device__ __forceinline__ void st16(u32x4 v, __amdgpu_buffer_rsrc_t r, unsigned off) {
    __builtin_amdgcn_raw_buffer_store_b128(v, r, off, 0, AUX_SYS);
}
template <int N> __device__ __forceinline__ void wait_vm() { asm volatile("s_waitcnt vmcnt(%0)" :: "n"(N) : "memory"); }

#define STAMP(idx) do {} while (0)
#define STAMP_FLUSH() do {} while (0)

__device__ __forceinline__ void load_w(const float* w, int ks, int quarter, int lane, f32x4 (&wr)[8]) {
    const float* wsrc = w + (size_t)(ks * 32 + (lane >> 4) * 8) * NSH + quarter * 64 + (lane & 15) * 4;
#pragma unroll
    for (int e = 0; e < 8; ++e) wr[e] = *(const f32x4*)(wsrc + e * NSH);
}
__device__ __forceinline__ u32x4 w_frag(const f32x4 (&wr)[8], int j) {
    return (u32x4){nz(pack2(wr[0][j], wr[1][j])), nz(pack2(wr[2][j], wr[3][j])), nz(pack2(wr[4][j], wr[5][j])), nz(pack2(wr[6][j], wr[7][j]))};
}
template <bool OPP>
__device__ __forceinline__ void send_all(const Params& p, int dsel, int u, int lane, unsigned c0, unsigned& call_out, char* lds, unsigned& stv) {
    constexpr int N0 = OPP ? NPSI : NPHI, FIRST = OPP ? NPHI : 0;
    constexpr int NT = N0 + NRHO, PF = PREF, WIN = OPP ? WINDOW_OPP : WINDOW;
    auto rt_of = [](int n) { return n < N0 ? FIRST + n : NPHI + NPSI + (n - N0); };
    auto ks_of = [&](int n) { return (u + n * ROT) & 63; };
    const float* xsrc = p.x + (size_t)(lane & 15) * KD + (lane >> 4) * 8;
    const int dest = (p.rank + dsel) & 7;
    LAS unsigned* tok = (LAS unsigned*)lds;
    f32x4 wr[8];
    f32x4 raw[NT > 0 ? NT : 1][2];
    auto ldx = [&](int i) {
        raw[i][0] = *(const f32x4*)(xsrc + (size_t)rt_of(i) * 16 * KD + ks_of(i) * 32);
        raw[i][1] = *(const f32x4*)(xsrc + (size_t)rt_of(i) * 16 * KD + ks_of(i) * 32 + 4);
    };
    if (OPP) {
        if (threadIdx.x == 0) *tok = 0;
        for (int i = 0; i < (u & 7); ++i) __builtin_amdgcn_s_sleep(WSTAG);
        load_w(p.w, u, dest >> 1, lane, wr);
        asm volatile("s_waitcnt lgkmcnt(0)" ::: "memory");
        __builtin_amdgcn_s_barrier();
    } else {
#pragma unroll
        for (int i = 0; i < PF && i < NT; ++i) ldx(i);
    }
    const unsigned call = __builtin_amdgcn_readfirstlane(c0) + 1;
    call_out = call;
    const unsigned so = (call % NSLOT) * SLOT_BYTES;
    char* dslot = p.peer[dsel] + so;
    auto rs = __builtin_amdgcn_make_buffer_rsrc(dslot + p.rank * SRC_BYTES, 0, SRC_BYTES, 0x00020000);
    if (OPP) {
        auto rw = __builtin_amdgcn_make_buffer_rsrc(dslot + W_OFF + (4 + (p.rank >> 1)) * WQ_BYTES, 0, WQ_BYTES, 0x00020000);
        u32x4 wv[4];
#pragma unroll
        for (int j = 0; j < 4; ++j) { wv[j] = w_frag(wr, j); asm volatile("" : "+v"(wv[j])); }
        STAMP(5);
        while ((int)__hip_atomic_load(tok, __ATOMIC_RELAXED, __HIP_MEMORY_SCOPE_WORKGROUP) + WGRP <= (u & 7)) __builtin_amdgcn_s_sleep(1);
#pragma unroll
        for (int j = 0; j < 4; ++j) st16(wv[j], rw, u * 4 * FRAG + lane * 16 + j * FRAG);
        STAMP(6);
        wait_vm<0>();
        if (lane == 0) __hip_atomic_fetch_add(tok, 1u, __ATOMIC_RELAXED, __HIP_MEMORY_SCOPE_WORKGROUP);
        STAMP(7);
#pragma unroll
        for (int i = 0; i < PF && i < NT; ++i) ldx(i);
        while (__hip_atomic_load(tok, __ATOMIC_RELAXED, __HIP_MEMORY_SCOPE_WORKGROUP) < (unsigned)XACK) __builtin_amdgcn_s_sleep(1);
    }
#pragma unroll
    for (int n = 0; n < NT; ++n) {
        const f32x4 r0 = raw[n][0], r1 = raw[n][1];
        const u32x4 v = {nz(pack2(r0.x, r0.y)), nz(pack2(r0.z, r0.w)), nz(pack2(r1.x, r1.y)), nz(pack2(r1.z, r1.w))};
        if (n >= WIN) {
            int young = 0;
            for (int q = n - WIN; q < n; ++q) { if (q > n - WIN) young += 1; if (q + PF < NT) young += 2; }
            switch (young) {
#define WCASE(k) case k: wait_vm<k>(); break;
                WCASE(0) WCASE(1) WCASE(2) WCASE(3) WCASE(4) WCASE(5) WCASE(6) WCASE(7) WCASE(8) WCASE(9) WCASE(10) WCASE(11) WCASE(12)
                default: break;
            }
        }
        if (n == 0) STAMP(8);
        st16(v, rs, lane * 16 + rt_of(n) * RT_BYTES + ks_of(n) * FRAG);
        if (n + PF < NT) ldx(n + PF);
        __builtin_amdgcn_sched_barrier(0);
    }
}

__global__ void __launch_bounds__(NTHREADS) ag_gemm_silu(Params p) {
    extern __shared__ __attribute__((aligned(16))) char lds[];
    const int b = blockIdx.x;
    const int wave = __builtin_amdgcn_readfirstlane(threadIdx.x >> 6);
    const int lane = threadIdx.x & 63;
    char* const myws = p.myws;
    unsigned* ctr = (unsigned*)(myws + CTR_OFF + b * 128) + wave;
    const u64 t0 = __builtin_amdgcn_s_memrealtime();
    unsigned stv = 0; (void)stv;
    STAMP(0);
    const unsigned c0 = __hip_atomic_load((GAS unsigned*)ctr, __ATOMIC_RELAXED, __HIP_MEMORY_SCOPE_AGENT);
    if (b < NSENDWG) {
        const int dsel = b & 7, u = (b >> 3) * 8 + wave;
        unsigned call;
        STAMP(1);
        if (COOP && (dsel & 1)) {
            send_all<true>(p, dsel, u, lane, c0, call, lds, stv);
            STAMP(2);

            if (lane == 0) *ctr = call;
        } else if (dsel != 0) {
            send_all<false>(p, dsel, u, lane, c0, call, lds, stv);
            STAMP(2);
            if (lane == 0) *ctr = call;
        } else {
            call = __builtin_amdgcn_readfirstlane(c0) + 1;
            if (lane == 0) *ctr = call;
            auto rz = __builtin_amdgcn_make_buffer_rsrc(myws + ((call + NSLOT - 1) % NSLOT) * SLOT_BYTES, 0, SLOT_BYTES, 0x00020000);
            const u32x4 z = {0, 0, 0, 0};
            constexpr int NW_ = COOP ? 32 : 16, NS_ = NPHI + NRHO, NO_ = NPSI + NRHO;
            const unsigned zo = u * FRAG + lane * 16;
            while (__builtin_amdgcn_s_memrealtime() - t0 < ZDELAY) __builtin_amdgcn_s_sleep(4);
            if (ZMODE != 1) for (int i = 0; i < NW_; ++i) st16(z, rz, zo + (128 + i) * RT_BYTES);
            if (ZMODE == 0) for (int k = 0; k < 4; ++k) {
                for (int i = 0; i < NS_; ++i) st16(z, rz, zo + ((2 * k + (p.rank & 1)) * 16 + (i < NPHI ? i : i + NPSI)) * RT_BYTES);
                for (int i = 0; i < NO_; ++i) st16(z, rz, zo + ((2 * k + 1 - (p.rank & 1)) * 16 + NPHI + i) * RT_BYTES);
            }
            STAMP(2);
            if (COOP && b == 0 && wave == 0) {
                const GAS unsigned* fl = (const GAS unsigned*)(myws + FLAG_OFF);
                for (;;) {
                    unsigned f0 = lane < NFLAGS ? __hip_atomic_load(fl + lane, __ATOMIC_RELAXED, __HIP_MEMORY_SCOPE_SYSTEM) : call;
                    unsigned f1 = lane + 64 < NFLAGS ? __hip_atomic_load(fl + lane + 64, __ATOMIC_RELAXED, __HIP_MEMORY_SCOPE_SYSTEM) : call;
                    if (__builtin_amdgcn_ballot_w64(f0 < call || f1 < call) == 0) break;
                    if (__builtin_amdgcn_s_memrealtime() - t0 > DEADLINE_TICKS) {
                        if (lane == 0) {
                            __hip_atomic_store((GAS unsigned*)(myws + TMO_OFF), call, __ATOMIC_RELAXED, __HIP_MEMORY_SCOPE_AGENT);
                            p.outq[0][(size_t)(p.rank * MSH) * NSH] = __builtin_nanf("");
                        }
                        break;
                    }
                    __builtin_amdgcn_s_sleep(2);
                }
                STAMP(4);
            }
        }
        return;
    }
    const int c = b - NSENDWG;
    const u64 task = p.task[c];
    const int q = task & 15, ntile = (task >> 4) & 15;
    LAS unsigned* cnt = (LAS unsigned*)(lds + MAXT * 32 * 1024);
    if (threadIdx.x < 8) cnt[threadIdx.x] = 0;
    bool failed = false;
    u32x4 B[8][4];
    const int unit = wave == 0 ? c : (wave == 1 && c < 256 - NCONS ? NCONS + c : -1);
    f32x4 wr[8];
    if (unit >= 0) load_w(p.w, unit & 63, unit >> 6, lane, wr);
    const int xi = wave >= 2 ? (wave - 2) * NCONS + c : 1 << 20;
    const int xn = xi >> 6, xrt = xn < NPHI ? xn : xn + NPSI;
    if (xi < (NPHI + NRHO) * 64) {
        const float* xs = p.x + (size_t)(xrt * 16 + (lane & 15)) * KD + (xi & 63) * 32 + (lane >> 4) * 8;
        wr[0] = *(const f32x4*)xs; wr[1] = *(const f32x4*)(xs + 4);
    }
    asm volatile("s_waitcnt lgkmcnt(0)" ::: "memory");
    __builtin_amdgcn_s_barrier();
    const unsigned call = __builtin_amdgcn_readfirstlane(c0) + 1;
    const unsigned so = (call % NSLOT) * SLOT_BYTES;
    if (unit >= 0) {
        auto rl = __builtin_amdgcn_make_buffer_rsrc(myws + so + W_OFF, 0, 4 * WQ_BYTES, 0x00020000);
#pragma unroll
        for (int j = 0; j < 4; ++j) st16(w_frag(wr, j), rl, (lane * 16 + j * FRAG) + (unit * 4 * FRAG));
    }
    if (xi < (NPHI + NRHO) * 64) {
        auto rl = __builtin_amdgcn_make_buffer_rsrc(myws + so + p.rank * SRC_BYTES, 0, SRC_BYTES, 0x00020000);
        st16((u32x4){nz(pack2(wr[0].x, wr[0].y)), nz(pack2(wr[0].z, wr[0].w)), nz(pack2(wr[1].x, wr[1].y)), nz(pack2(wr[1].z, wr[1].w))},
             rl, xrt * RT_BYTES + (xi & 63) * FRAG + lane * 16);
    }
    if (lane == 0) __hip_atomic_store((GAS unsigned*)ctr, call, __ATOMIC_RELAXED, __HIP_MEMORY_SCOPE_AGENT);
    auto ra = __builtin_amdgcn_make_buffer_rsrc(myws + so, 0, W_OFF, 0x00020000);
    const unsigned ao = wave * FRAG + lane * 16;
    u32x4 a[8];
    unsigned have = 0;
    {
        auto rw = __builtin_amdgcn_make_buffer_rsrc(myws + so + W_OFF + q * WQ_BYTES, 0, WQ_BYTES, 0x00020000);
        const unsigned wo = wave * 4 * FRAG + lane * 16;
        const unsigned tl0 = (task >> 16) & 127;
        for (;;) {
            asm volatile("" ::: "memory");
            unsigned cv = 1;
            if (lane < 16) cv = __builtin_amdgcn_raw_buffer_load_b32(rw, wave * 4 * FRAG + 960 + lane * 4, (BCAN_I * 32 + 3) * FRAG, AUX_SYS);
            unsigned want = q < 4 ? 0u : (have == 0 ? 1u : (~have & 0xFFu));
#pragma unroll
            for (int i = 0; i < 8; ++i)
                if (want & (1u << i)) a[i] = __builtin_amdgcn_raw_buffer_load_b128(ra, ao + i * 8 * FRAG, tl0 * RT_BYTES, AUX_SYS);
            asm volatile("" : "+s"(want));
#pragma unroll
            for (int i = 0; i < 8; ++i)
                if (want & (1u << i)) { if (full(a[i])) have |= 1u << i; }
            __builtin_amdgcn_s_waitcnt(0x0F70);
            if (__builtin_amdgcn_ballot_w64(cv == 0) == 0) break;
            if (failed || __builtin_amdgcn_s_memrealtime() - t0 > DEADLINE_TICKS) { failed = true; break; }
            __builtin_amdgcn_s_sleep(B_SLEEP);
        }
        STAMP(2);
        unsigned pend = 0xFFFFFFFFu;
        {
#pragma unroll
            for (int f = 0; f < 32; ++f) B[f >> 2][f & 3] = __builtin_amdgcn_raw_buffer_load_b128(rw, wo, ((f >> 2) * 32 + (f & 3)) * FRAG, AUX_SYS);
            unsigned mn = ~0u;
#pragma unroll
            for (int f = 0; f < 32; ++f) { const u32x4 v = B[f >> 2][f & 3]; mn = min(min(mn, v.x), v.y); mn = min(min(mn, v.z), v.w); }
            if (__builtin_amdgcn_ballot_w64(mn == 0) == 0) pend = 0;
            else {
#pragma unroll
                for (int f = 0; f < 32; ++f) if (full(B[f >> 2][f & 3])) pend &= ~(1u << f);
            }
        }
        while (pend) {
            asm volatile("" ::: "memory");
            unsigned want = pend;
#pragma unroll
            for (int f = 0; f < 32; ++f)
                if (want >> f & 1) B[f >> 2][f & 3] = __builtin_amdgcn_raw_buffer_load_b128(rw, wo, ((f >> 2) * 32 + (f & 3)) * FRAG, AUX_SYS);
            asm volatile("" : "+s"(want));
#pragma unroll
            for (int f = 0; f < 32; ++f)
                if (want >> f & 1) { if (full(B[f >> 2][f & 3])) pend &= ~(1u << f); }
            __builtin_amdgcn_s_waitcnt(0x0F70);
            if (pend == want) {
                if (failed || __builtin_amdgcn_s_memrealtime() - t0 > DEADLINE_TICKS) { failed = true; break; }
                __builtin_amdgcn_s_sleep(B_SLEEP);
            }
        }
    }
    STAMP(1);
    float* const outp = p.outq[q];
    for (int t = 0; t < ntile; ++t) {
        const unsigned tl = (task >> (16 + 7 * t)) & 127;
        f32x4 acc[4] = {};
        unsigned pending = 0xFF;
        for (;;) {
#pragma unroll
            for (int i = 0; i < 8; ++i) {
                if (have & (1u << i)) {
#pragma unroll
                    for (int j = 0; j < 4; ++j)
                        acc[j] = __builtin_amdgcn_mfma_f32_16x16x32_f16(__builtin_bit_cast(half8, a[i]), __builtin_bit_cast(half8, B[i][j]), acc[j], 0, 0, 0);
                }
            }
            pending &= ~have;
            if (!pending) break;
            if (!have) {
                if (failed || __builtin_amdgcn_s_memrealtime() - t0 > DEADLINE_TICKS) { failed = true; break; }
                __builtin_amdgcn_s_sleep(POLL_SLEEP);
            }
            asm volatile("" ::: "memory");
            unsigned want = pending == 0xFF ? 1u : pending;
#pragma unroll
            for (int i = 0; i < 8; ++i)
                if (want & (1u << i)) a[i] = __builtin_amdgcn_raw_buffer_load_b128(ra, ao + i * 8 * FRAG, tl * RT_BYTES, AUX_SYS);
            asm volatile("" : "+s"(want));
            have = 0;
#pragma unroll
            for (int i = 0; i < 8; ++i)
                if (want & (1u << i)) { if (full(a[i])) have |= 1u << i; }
            __builtin_amdgcn_s_waitcnt(0x0F70);
        }
        STAMP(8 + t * 2);
        const bool more = t + 1 < ntile;
        if (more) {
            const unsigned tn = (task >> (23 + 7 * t)) & 127;
#pragma unroll
            for (int i = 0; i < 8; ++i) a[i] = __builtin_amdgcn_raw_buffer_load_b128(ra, ao + i * 8 * FRAG, tn * RT_BYTES, AUX_SYS);
        }
        char* part = lds + t * 32 * 1024;
        const int po = (lane >> 4) * 1024 + (lane & 15) * 16;
#pragma unroll
        for (int r = 0; r < 4; ++r) {
            f32x4 v = {acc[0][r], acc[1][r], acc[2][r], acc[3][r]};
            if (failed) v = (f32x4){__builtin_nanf(""), __builtin_nanf(""), __builtin_nanf(""), __builtin_nanf("")};
            *(f32x4*)(part + wave * 4096 + po + r * 256) = v;
        }
        asm volatile("s_waitcnt lgkmcnt(0)" ::: "memory");
        unsigned old = 0;
        if (lane == 0) old = __hip_atomic_fetch_add(cnt + t, 1u, __ATOMIC_RELAXED, __HIP_MEMORY_SCOPE_WORKGROUP);
        old = __builtin_amdgcn_readfirstlane(old);
        auto check = [&]() {
            have = 0;
#pragma unroll
            for (int i = 0; i < 8; ++i) if (full(a[i])) have |= 1u << i;
        };
        if (!more) {
            while (__hip_atomic_load(cnt + t, __ATOMIC_RELAXED, __HIP_MEMORY_SCOPE_WORKGROUP) < 8u) {}
            asm volatile("" ::: "memory");
            f32x2 s = *(const f32x2*)(part + wave * 512 + lane * 8);
#pragma unroll
            for (int ww = 1; ww < 8; ++ww) s += *(const f32x2*)(part + ww * 4096 + wave * 512 + lane * 8);
            f32x2 o;
#pragma unroll
            for (int e = 0; e < 2; ++e) o[e] = s[e] * __builtin_amdgcn_rcpf(1.f + __expf(-fminf(fmaxf(s[e], -60.f), 60.f)));
            float* op = outp + (size_t)(tl * 16 + wave * 2 + (lane >> 5)) * NSH + (lane & 31) * 2;
            __hip_atomic_store((GAS u64*)op, __builtin_bit_cast(u64, o), __ATOMIC_RELAXED, __HIP_MEMORY_SCOPE_SYSTEM);
            STAMP(9 + t * 2);
        } else if (old == 7) {
            asm volatile("" ::: "memory");
#pragma unroll
            for (int r = 0; r < 4; ++r) {
                f32x4 s = *(const f32x4*)(part + r * 1024 + lane * 16);
#pragma unroll
                for (int ww = 1; ww < 8; ++ww) s += *(const f32x4*)(part + ww * 4096 + r * 1024 + lane * 16);
                f32x4 o;
#pragma unroll
                for (int e = 0; e < 4; ++e) o[e] = s[e] * __builtin_amdgcn_rcpf(1.f + __expf(-fminf(fmaxf(s[e], -60.f), 60.f)));
                auto ro = __builtin_amdgcn_make_buffer_rsrc(outp, 0, (2048 * NSH - 192) * 4, 0x00020000);
                st16(__builtin_bit_cast(u32x4, o), ro, (((lane >> 4) * NSH + (lane & 15) * 4) * 4 + r * 4 * NSH * 4) + (tl * 16 * NSH * 4));
            }
            STAMP(9 + t * 2);
            check();
        } else check();
    }
    if (failed && lane == 0) __hip_atomic_store((GAS unsigned*)(myws + TMO_OFF), call, __ATOMIC_RELAXED, __HIP_MEMORY_SCOPE_AGENT);
    if (q >= 4) {
        wait_vm<0>();
        if (lane == 0 && __hip_atomic_fetch_add(cnt + 7, 1u, __ATOMIC_RELAXED, __HIP_MEMORY_SCOPE_WORKGROUP) == 7)
            __hip_atomic_store((GAS unsigned*)p.flagq[q - 4] + ((task >> 8) & 255), call, __ATOMIC_RELAXED, __HIP_MEMORY_SCOPE_SYSTEM);
    }
    STAMP(3);
    STAMP_FLUSH();
    const int zi = (task >> 48) & 255;
    if (ZMODE != 0 && zi < NZWG) {
        auto rz = __builtin_amdgcn_make_buffer_rsrc(myws + ((call + NSLOT - 1) % NSLOT) * SLOT_BYTES, 0, SLOT_BYTES, 0x00020000);
        const u32x4 z = {0, 0, 0, 0};
        const int wi = zi * 8 + wave;
        constexpr int NW_ = COOP ? 32 : 16, NS_ = NPHI + NRHO, NO_ = NPSI + NRHO;
        for (int i = (ZMODE == 2 ? NW_ : 0) + (wi >> 6); i < NZ; i += NZWG / 8) {
            int unit = 128 + i;
            if (i >= NW_) {
                const int k = (i - NW_) / (NS_ + NO_), rem = (i - NW_) % (NS_ + NO_);
                unit = rem < NS_ ? (2 * k + (p.rank & 1)) * 16 + (rem < NPHI ? rem : rem + NPSI)
                                 : (2 * k + 1 - (p.rank & 1)) * 16 + NPHI + (rem - NS_);
            }
            st16(z, rz, (wi & 63) * FRAG + lane * 16 + unit * RT_BYTES);
        }
    }
}

static void build_tasks(int rank, u64* task) {
    struct RT { double t; int id; bool coop; };
    std::vector<RT> rts;
    for (int s = 0; s < NR; ++s) {
        const bool same = ((s ^ rank) & 1) == 0;
        const int n0 = same ? NPHI : NPSI, first = same ? 0 : NPHI;
        for (int i = 0; i < n0 + NRHO; ++i) {
            const int rt = i < n0 ? first + i : NPHI + NPSI + (i - n0);
            double t = s == rank ? 0.05 * (i + 1) : (same || !COOP) ? (i + 1) : OPP_SCALE * (4 + i + 1);
            rts.push_back({t + 0.001 * ((s - rank) & 7), s * 16 + rt, i < n0});
        }
    }
    std::stable_sort(rts.begin(), rts.end(), [](const RT& a, const RT& b) { return a.t < b.t; });
    std::vector<std::pair<double, int>> fin;
    std::vector<RT> own[4];
    { int n = 0; for (auto& r : rts) own[n++ % 4].push_back(r); }
    int nown = 0;
    for (int X = 0; X < 8; ++X) {
        std::vector<int> lists[24]; double last[24] = {}; int qq[24];
        if (X & 1) {
            const int j = (rank + X) & 7;
            int k = 0;
            for (auto& r : rts) if (r.coop) { last[k % 24] = r.t; lists[k++ % 24].push_back(r.id); }
            for (int i = 0; i < 24; ++i) qq[i] = 4 + (j >> 1);
        } else {
            for (int q = 0; q < 4; ++q) {
                int k = q;
                for (auto& r : own[X >> 1]) { last[q * 6 + k % 6] = r.t; lists[q * 6 + k++ % 6].push_back(r.id); }
                for (int i = 0; i < 6; ++i) qq[q * 6 + i] = q;
            }
        }
        for (int i = 0; i < 24; ++i) {
            u64 e = (u64)qq[i] | ((u64)lists[i].size() << 4) | ((u64)((X & 1) ? i : nown++) << 8);
            if (lists[i].size() > (size_t)MAXT) { fprintf(stderr, "too many tiles\n"); abort(); }
            for (size_t t = 0; t < lists[i].size(); ++t) e |= (u64)lists[i][t] << (16 + 7 * t);
            task[X + 8 * i] = e | (255ull << 48);
            fin.push_back({last[i] + ((X & 1) ? 1.5 : 0.0), X + 8 * i});
        }
    }
    std::stable_sort(fin.begin(), fin.end());
    for (int i = 0; i < NZWG; ++i) task[fin[i].second] = (task[fin[i].second] & ~(255ull << 48)) | ((u64)i << 48);
}

extern "C" void kernel_launch(int rank, int nranks, ncclComm_t comm, hipStream_t stream,
                              void* const* d_in, const int* in_sizes, int n_in,
                              void* d_out, int out_size, const int* in_dtypes, int out_dtype,
                              void* const* d_out_all, void* const* d_ws_all, size_t ws_bytes) {
    if (nranks != NR || ws_bytes < WS_NEED || (unsigned)rank >= (unsigned)NR) { fprintf(stderr, "bad config\n"); return; }
    HIP_CHECK(hipFuncSetAttribute((const void*)ag_gemm_silu, hipFuncAttributeMaxDynamicSharedMemorySize, LDS_BYTES));
    Params p;
    p.x = (const float*)d_in[0]; p.w = (const float*)d_in[1];
    p.myws = (char*)d_ws_all[rank];
    for (int i = 0; i < NR; ++i) p.peer[i] = (char*)d_ws_all[(rank + i) % NR];
    for (int q = 0; q < 4; ++q) {
        const int j = q * 2 + ((rank & 1) ^ 1);
        p.outq[q] = (float*)d_out + q * 64;
        p.outq[4 + q] = (float*)d_out_all[j] + (rank >> 1) * 64;
        p.flagq[q] = (unsigned*)((char*)d_ws_all[j] + FLAG_OFF) + (rank >> 1) * 8 * (6 - OWNWG);
    }
    p.rank = rank; p.pad = 0;
    build_tasks(rank, p.task);
    hipLaunchKernelGGL(ag_gemm_silu, dim3(NWG), dim3(NTHREADS), LDS_BYTES, stream, p);
    HIP_CHECK(hipGetLastError());
}
```

```cpp
#include <hip/hip_runtime.h>
#include <rccl/rccl.h>
#include <cstdio>
#include <vector>
#include <algorithm>

#define HIP_CHECK(expr) do { hipError_t _e = (expr); if (_e != hipSuccess) { \
    fprintf(stderr, "HIP error %s at %s:%d\n", hipGetErrorString(_e), __FILE__, __LINE__); \
    return; } } while (0)

#define GAS __attribute__((address_space(1)))
#define LAS __attribute__((address_space(3)))
typedef _Float16 half8 __attribute__((ext_vector_type(8)));
typedef _Float16 half2v __attribute__((ext_vector_type(2)));
typedef float f32x4 __attribute__((ext_vector_type(4)));
typedef float f32x2 __attribute__((ext_vector_type(2)));
typedef unsigned u32x4 __attribute__((ext_vector_type(4)));
typedef unsigned long long u64;

constexpr int NR = 8, MSH = 256, KD = 2048, NSH = 256;
constexpr int NPHI = 11, NPSI = 4, NRHO = 16 - NPHI - NPSI;
constexpr int NTHREADS = 512, NWG = 256, NSENDWG = 64, NCONS = NWG - NSENDWG, CPX = NCONS / 8;
constexpr int MAXT = 3;
constexpr int ROT = 9;
constexpr int WGRP = 4, XACK = 7;
constexpr int ZWIN = 4;
constexpr int AUX_SYS = 17;
constexpr float WSCALE = 32.f;
constexpr int XL = 4, XH = 16;

constexpr unsigned FRAG = 1024, RT_BYTES = 64 * FRAG, SRC_BYTES = 16 * RT_BYTES;
constexpr unsigned W_OFF = NR * SRC_BYTES;
constexpr unsigned WQ_BYTES = 256 * FRAG;
constexpr unsigned SLOT_BYTES = W_OFF + 8 * WQ_BYTES;
constexpr int NSLOT = 3;
constexpr unsigned CTR_OFF = NSLOT * SLOT_BYTES;
constexpr unsigned TMO_OFF = CTR_OFF + NWG * 128;
constexpr unsigned FLAG_OFF = TMO_OFF + 4096;
constexpr unsigned WS_NEED = FLAG_OFF + 4096;
constexpr int NFLAGS = 4 * CPX;
constexpr int NZ = 32 + 4 * (NPHI + NRHO) + 4 * (NPSI + NRHO);
constexpr u64 DEADLINE_TICKS = 150000000ull;
constexpr int LDS_BYTES = 128 * 1024 + 64;
static_assert(MAXT * 32 * 1024 <= 128 * 1024 && NFLAGS <= 128, "layout");

struct Params {
    const float* x; const float* w;
    int rank; int pad;
    char* peer[NR];
    float* outq[8];
    unsigned* flagq[4];
    u64 task[NCONS];
};

__device__ __forceinline__ unsigned pack2(float a, float b) {
    half2v h = __builtin_convertvector((f32x2){a, b}, half2v);
    return __builtin_bit_cast(unsigned, h);
}
__device__ __forceinline__ unsigned nz(unsigned v) { return v ? v : 0x8000u; }
__device__ __forceinline__ bool full(const u32x4& a) {
    const unsigned mn = min(min(a.x, a.y), min(a.z, a.w));
    return __builtin_amdgcn_ballot_w64(mn == 0) == 0;
}
__device__ __forceinline__ void st16(u32x4 v, __amdgpu_buffer_rsrc_t r, unsigned off) {
    __builtin_amdgcn_raw_buffer_store_b128(v, r, off, 0, AUX_SYS);
}
__device__ __forceinline__ u32x4 ld16(__amdgpu_buffer_rsrc_t r, unsigned voff, unsigned soff) {
    return __builtin_amdgcn_raw_buffer_load_b128(r, voff, soff, AUX_SYS);
}
__device__ __forceinline__ __amdgpu_buffer_rsrc_t rsrc(void* p, unsigned bytes) {
    return __builtin_amdgcn_make_buffer_rsrc(p, 0, bytes, 0x00020000);
}
__device__ __forceinline__ float silu(float y) {
    return y * __builtin_amdgcn_rcpf(1.f + __expf(-fminf(fmaxf(y, -60.f), 60.f)));
}
template <int N> __device__ __forceinline__ void wait_vm() { asm volatile("s_waitcnt vmcnt(%0)" :: "n"(N) : "memory"); }
__device__ __forceinline__ void vm_clean() { __builtin_amdgcn_s_waitcnt(0x0F70); }
__device__ __forceinline__ unsigned lds_load(LAS unsigned* p) { return __hip_atomic_load(p, __ATOMIC_RELAXED, __HIP_MEMORY_SCOPE_WORKGROUP); }
__device__ __forceinline__ unsigned lds_add(LAS unsigned* p) { return __hip_atomic_fetch_add(p, 1u, __ATOMIC_RELAXED, __HIP_MEMORY_SCOPE_WORKGROUP); }
__device__ __forceinline__ unsigned ctr_load(GAS unsigned* p) { return __hip_atomic_load(p, __ATOMIC_RELAXED, __HIP_MEMORY_SCOPE_AGENT); }
__device__ __forceinline__ void ctr_store(GAS unsigned* p, unsigned v) { __hip_atomic_store(p, v, __ATOMIC_RELAXED, __HIP_MEMORY_SCOPE_AGENT); }
__device__ __forceinline__ bool expired(u64 t0) { return __builtin_amdgcn_s_memrealtime() - t0 > DEADLINE_TICKS; }

__device__ __forceinline__ void load_w(const float* w, int ks, int quarter, int lane, f32x4 (&wr)[8]) {
    const float* s = w + (size_t)(ks * 32 + (lane >> 4) * XL) * NSH + quarter * 64 + (lane & 15) * 4;
#pragma unroll
    for (int e = 0; e < 8; ++e) wr[e] = *(const f32x4*)(s + ((e & 3) + (e >> 2) * XH) * NSH);
}
__device__ __forceinline__ u32x4 w_frag(const f32x4 (&wr)[8], int j) {
    return (u32x4){nz(pack2(wr[0][j] * WSCALE, wr[1][j] * WSCALE)), nz(pack2(wr[2][j] * WSCALE, wr[3][j] * WSCALE)),
                   nz(pack2(wr[4][j] * WSCALE, wr[5][j] * WSCALE)), nz(pack2(wr[6][j] * WSCALE, wr[7][j] * WSCALE))};
}
__device__ __forceinline__ void load_x(const float* x, int rt, int ks, int lane, f32x4 (&r)[2]) {
    const float* s = x + (size_t)(rt * 16 + (lane & 15)) * KD + ks * 32 + (lane >> 4) * XL;
    r[0] = *(const f32x4*)s; r[1] = *(const f32x4*)(s + XH);
}
__device__ __forceinline__ u32x4 x_frag(const f32x4 (&r)[2]) {
    return (u32x4){nz(pack2(r[0].x, r[0].y)), nz(pack2(r[0].z, r[0].w)), nz(pack2(r[1].x, r[1].y)), nz(pack2(r[1].z, r[1].w))};
}

template <bool OPP>
__device__ __forceinline__ void send_all(const Params& p, int dsel, int u, int lane, GAS unsigned* ctr, char* lds) {
    constexpr int N0 = OPP ? NPSI : NPHI, FIRST = OPP ? NPHI : 0, NT = N0 + NRHO;
    auto rt_of = [](int n) { return n < N0 ? FIRST + n : NPHI + NPSI + (n - N0); };
    auto ks_of = [&](int n) { return (u + n * ROT) & 63; };
    const int wave = u & 7, dest = (p.rank + dsel) & 7;
    LAS unsigned* acked = (LAS unsigned*)lds;
    f32x4 wr[8], raw[NT][2];
    if (OPP) {
        if (threadIdx.x == 0) *acked = 0;
        for (int i = 0; i < wave; ++i) __builtin_amdgcn_s_sleep(3);
        load_w(p.w, u, dest >> 1, lane, wr);
    } else load_x(p.x, rt_of(0), ks_of(0), lane, raw[0]);
    const unsigned c0 = ctr_load(ctr);
    if (OPP) {
        asm volatile("s_waitcnt lgkmcnt(0)" ::: "memory");
        __builtin_amdgcn_s_barrier();
    } else load_x(p.x, rt_of(1), ks_of(1), lane, raw[1]);
    const unsigned call = __builtin_amdgcn_readfirstlane(c0) + 1;
    char* dslot = p.peer[dsel] + (call % NSLOT) * SLOT_BYTES;
    auto rs = rsrc(dslot + p.rank * SRC_BYTES, SRC_BYTES);
    if (OPP) {
        auto rw = rsrc(dslot + W_OFF + (4 + (p.rank >> 1)) * WQ_BYTES, WQ_BYTES);
        u32x4 wv[4];
#pragma unroll
        for (int j = 0; j < 4; ++j) { wv[j] = w_frag(wr, j); asm volatile("" : "+v"(wv[j])); }
        while ((int)lds_load(acked) + WGRP <= wave) __builtin_amdgcn_s_sleep(1);
#pragma unroll
        for (int j = 0; j < 4; ++j) st16(wv[j], rw, u * 4 * FRAG + lane * 16 + j * FRAG);
        wait_vm<0>();
        if (lane == 0) lds_add(acked);
        load_x(p.x, rt_of(0), ks_of(0), lane, raw[0]);
        load_x(p.x, rt_of(1), ks_of(1), lane, raw[1]);
        while (lds_load(acked) < (unsigned)XACK) __builtin_amdgcn_s_sleep(1);
    }
#pragma unroll
    for (int n = 0; n < NT; ++n) {
        st16(x_frag(raw[n]), rs, lane * 16 + rt_of(n) * RT_BYTES + ks_of(n) * FRAG);
        if (n + 2 < NT) load_x(p.x, rt_of(n + 2), ks_of(n + 2), lane, raw[n + 2]);
        __builtin_amdgcn_sched_barrier(0);
    }
    if (lane == 0) ctr_store(ctr, call);
}

__device__ __forceinline__ int zero_unit(int i, int rank) {
    constexpr int NS_ = NPHI + NRHO, NO_ = NPSI + NRHO;
    if (i < 32) return 128 + i;
    const int k = (i - 32) / (NS_ + NO_), rem = (i - 32) % (NS_ + NO_);
    return rem < NS_ ? (2 * k + (rank & 1)) * 16 + (rem < NPHI ? rem : rem + NPSI)
                     : (2 * k + 1 - (rank & 1)) * 16 + NPHI + (rem - NS_);
}

__global__ void __launch_bounds__(NTHREADS) ag_gemm_silu(Params p) {
    extern __shared__ __attribute__((aligned(16))) char lds[];
    const int b = blockIdx.x;
    const int wave = __builtin_amdgcn_readfirstlane(threadIdx.x >> 6);
    const int lane = threadIdx.x & 63;
    char* const myws = p.peer[0];
    GAS unsigned* ctr = (GAS unsigned*)(myws + CTR_OFF + b * 128) + wave;
    if (__builtin_expect(b < NSENDWG && (b & 7) && !(b & 1), 1)) {
        send_all<false>(p, b & 7, (b >> 3) * 8 + wave, lane, ctr, lds);
        return;
    }
    const u64 t0 = __builtin_amdgcn_s_memrealtime();
    if (b < NSENDWG) {
        const int dsel = b & 7, u = (b >> 3) * 8 + wave;
        if (dsel) send_all<true>(p, dsel, u, lane, ctr, lds);
        else {
            const unsigned call = __builtin_amdgcn_readfirstlane(ctr_load(ctr)) + 1;
            if (lane == 0) ctr_store(ctr, call);
            auto rz = rsrc(myws + ((call + NSLOT - 1) % NSLOT) * SLOT_BYTES, SLOT_BYTES);
            for (int i = 0; i < NZ; ++i) {
                wait_vm<ZWIN - 1>();
                st16((u32x4){0, 0, 0, 0}, rz, u * FRAG + lane * 16 + zero_unit(i, p.rank) * RT_BYTES);
            }
            if (u == 0) {
                const GAS unsigned* fl = (const GAS unsigned*)(myws + FLAG_OFF);
                for (;;) {
                    const unsigned f0 = __hip_atomic_load(fl + lane, __ATOMIC_RELAXED, __HIP_MEMORY_SCOPE_SYSTEM);
                    const unsigned f1 = lane + 64 < NFLAGS ? __hip_atomic_load(fl + lane + 64, __ATOMIC_RELAXED, __HIP_MEMORY_SCOPE_SYSTEM) : call;
                    if (__builtin_amdgcn_ballot_w64(f0 < call || f1 < call) == 0) break;
                    if (expired(t0)) {
                        if (lane == 0) {
                            __hip_atomic_store((GAS unsigned*)(myws + TMO_OFF), call, __ATOMIC_RELAXED, __HIP_MEMORY_SCOPE_AGENT);
                            p.outq[0][(size_t)(p.rank * MSH) * NSH] = __builtin_nanf("");
                        }
                        break;
                    }
                    __builtin_amdgcn_s_sleep(1);
                }
            }
        }
        return;
    }

    const int c = b - NSENDWG;
    const u64 task = p.task[c];
    const int q = task & 7, ntile = (task >> 4) & 3;
    LAS unsigned* cnt = (LAS unsigned*)(lds + 128 * 1024);
    if (threadIdx.x < 8) cnt[threadIdx.x] = 0;
    const int unit = wave == 0 ? c : (wave == 1 && c < 256 - NCONS ? NCONS + c : -1);
    const int xi = wave >= 2 ? (wave - 2) * NCONS + c : 1 << 20;
    const bool xconv = xi < (NPHI + NRHO) * 64;
    const int xrt = (xi >> 6) < NPHI ? (xi >> 6) : (xi >> 6) + NPSI;
    f32x4 wr[8], xr[2];
    if (unit >= 0) load_w(p.w, unit & 63, unit >> 6, lane, wr);
    if (xconv) load_x(p.x, xrt, xi & 63, lane, xr);
    const unsigned c0 = ctr_load(ctr);
    asm volatile("s_waitcnt lgkmcnt(0)" ::: "memory");
    __builtin_amdgcn_s_barrier();
    const unsigned call = __builtin_amdgcn_readfirstlane(c0) + 1;
    char* const slot = myws + (call % NSLOT) * SLOT_BYTES;
    if (unit >= 0) {
        auto rl = rsrc(slot + W_OFF, 4 * WQ_BYTES);
#pragma unroll
        for (int j = 0; j < 4; ++j) st16(w_frag(wr, j), rl, (unit * 4 + j) * FRAG + lane * 16);
    }
    if (xconv) st16(x_frag(xr), rsrc(slot + p.rank * SRC_BYTES, SRC_BYTES), xrt * RT_BYTES + (xi & 63) * FRAG + lane * 16);
    if (lane == 0) ctr_store(ctr, call);

    auto ra = rsrc(slot, W_OFF);
    const unsigned ao = wave * FRAG + lane * 16;
    bool failed = false;
    u32x4 B[8][4], a[8];
    unsigned have = 0;
    {
        auto rw = rsrc(slot + W_OFF + q * WQ_BYTES, WQ_BYTES);
        const unsigned wo = wave * 4 * FRAG + lane * 16, tl0 = (task >> 16) & 127;
        for (;;) {
            asm volatile("" ::: "memory");
            unsigned cv = 1;
            if (lane < 16) cv = __builtin_amdgcn_raw_buffer_load_b32(rw, wave * 4 * FRAG + 960 + lane * 4, (7 * 32 + 3) * FRAG, AUX_SYS);
            unsigned want = q < 4 ? 0u : (have == 0 ? 1u : (~have & 0xFFu));
#pragma unroll
            for (int i = 0; i < 8; ++i) if (want & (1u << i)) a[i] = ld16(ra, ao + i * 8 * FRAG, tl0 * RT_BYTES);
            asm volatile("" : "+s"(want));
#pragma unroll
            for (int i = 0; i < 8; ++i) if (want & (1u << i)) { if (full(a[i])) have |= 1u << i; }
            vm_clean();
            if (__builtin_amdgcn_ballot_w64(cv == 0) == 0) break;
            if (expired(t0)) { failed = true; break; }
            __builtin_amdgcn_s_sleep(2);
        }
        unsigned pend = 0xFFFFFFFFu;
#pragma unroll
        for (int f = 0; f < 32; ++f) B[f >> 2][f & 3] = ld16(rw, wo, ((f >> 2) * 32 + (f & 3)) * FRAG);
        unsigned mn = ~0u;
#pragma unroll
        for (int f = 0; f < 32; ++f) { const u32x4 v = B[f >> 2][f & 3]; mn = min(min(mn, v.x), v.y); mn = min(min(mn, v.z), v.w); }
        if (__builtin_amdgcn_ballot_w64(mn == 0) == 0) pend = 0;
        else {
#pragma unroll
            for (int f = 0; f < 32; ++f) if (full(B[f >> 2][f & 3])) pend &= ~(1u << f);
        }
        while (pend) {
            asm volatile("" ::: "memory");
            unsigned want = pend;
#pragma unroll
            for (int f = 0; f < 32; ++f) if (want >> f & 1) B[f >> 2][f & 3] = ld16(rw, wo, ((f >> 2) * 32 + (f & 3)) * FRAG);
            asm volatile("" : "+s"(want));
#pragma unroll
            for (int f = 0; f < 32; ++f) if (want >> f & 1) { if (full(B[f >> 2][f & 3])) pend &= ~(1u << f); }
            vm_clean();
            if (pend == want) {
                if (failed || expired(t0)) { failed = true; break; }
                __builtin_amdgcn_s_sleep(2);
            }
        }
    }
    float* const outp = p.outq[q];
    for (int t = 0; t < ntile; ++t) {
        const unsigned tl = (task >> (16 + 7 * t)) & 127;
        f32x4 acc[4] = {};
        unsigned pending = 0xFF;
        const unsigned rtl = tl & 15, sl = tl >> 4;
        const bool samep = ((sl ^ p.rank) & 1) == 0;
        const bool haspred = sl != (unsigned)p.rank && rtl != (samep ? 0u : (unsigned)NPHI);
        const unsigned pl = (rtl == NPHI + NPSI && samep) ? tl - NPSI - 1 : tl - 1;
        bool armed = false;
        for (;;) {
#pragma unroll
            for (int i = 0; i < 8; ++i) {
                if (have & (1u << i)) {
#pragma unroll
                    for (int j = 0; j < 4; ++j)
                        acc[j] = __builtin_amdgcn_mfma_f32_16x16x32_f16(__builtin_bit_cast(half8, a[i]), __builtin_bit_cast(half8, B[i][j]), acc[j], 0, 0, 0);
                }
            }
            pending &= ~have;
            if (!pending) break;
            if (!have) {
                if (failed || expired(t0)) { failed = true; break; }
                __builtin_amdgcn_s_sleep(2);
            }
            asm volatile("" ::: "memory");
            have = 0;
            if (haspred && !armed && pending == 0xFF) {
                unsigned cv = 1;
                if (lane < 16) cv = __builtin_amdgcn_raw_buffer_load_b32(ra, wave * FRAG + 960 + lane * 4, pl * RT_BYTES, AUX_SYS);
                armed = __builtin_amdgcn_ballot_w64(cv == 0) == 0;
                vm_clean();
                if (!armed) continue;
            }
            unsigned want = (pending == 0xFF && !armed) ? 1u : pending;
#pragma unroll
            for (int i = 0; i < 8; ++i) if (want & (1u << i)) a[i] = ld16(ra, ao + i * 8 * FRAG, tl * RT_BYTES);
            asm volatile("" : "+s"(want));
#pragma unroll
            for (int i = 0; i < 8; ++i) if (want & (1u << i)) { if (full(a[i])) have |= 1u << i; }
            vm_clean();
        }
        const bool more = t + 1 < ntile;
        if (more) {
            const unsigned tn = (task >> (23 + 7 * t)) & 127;
#pragma unroll
            for (int i = 0; i < 8; ++i) a[i] = ld16(ra, ao + i * 8 * FRAG, tn * RT_BYTES);
        }
        char* part = lds + t * 32 * 1024;
#pragma unroll
        for (int r = 0; r < 4; ++r) {
            f32x4 v = {acc[0][r], acc[1][r], acc[2][r], acc[3][r]};
            if (failed) v = (f32x4){__builtin_nanf(""), __builtin_nanf(""), __builtin_nanf(""), __builtin_nanf("")};
            *(f32x4*)(part + wave * 4096 + ((lane >> 4) * 4 + r) * 256 + (lane & 15) * 16) = v;
        }
        asm volatile("s_waitcnt lgkmcnt(0)" ::: "memory");
        unsigned old = 0;
        if (lane == 0) old = lds_add(cnt + t);
        old = __builtin_amdgcn_readfirstlane(old);
        if (!more) {
            while (lds_load(cnt + t) < 8u) {}
            asm volatile("" ::: "memory");
            f32x2 s = *(const f32x2*)(part + wave * 512 + lane * 8);
#pragma unroll
            for (int ww = 1; ww < 8; ++ww) s += *(const f32x2*)(part + ww * 4096 + wave * 512 + lane * 8);
            const f32x2 o = {silu(s.x * (1.f / WSCALE)), silu(s.y * (1.f / WSCALE))};
            float* op = outp + (size_t)(tl * 16 + wave * 2 + (lane >> 5)) * NSH + (lane & 31) * 2;
            __hip_atomic_store((GAS u64*)op, __builtin_bit_cast(u64, o), __ATOMIC_RELAXED, __HIP_MEMORY_SCOPE_SYSTEM);
        } else {
            if (old == 7) {
                asm volatile("" ::: "memory");
                auto ro = rsrc(outp, (NR * MSH * NSH - 192) * 4);
#pragma unroll
                for (int r = 0; r < 4; ++r) {
                    f32x4 s = *(const f32x4*)(part + r * 1024 + lane * 16);
#pragma unroll
                    for (int ww = 1; ww < 8; ++ww) s += *(const f32x4*)(part + ww * 4096 + r * 1024 + lane * 16);
                    f32x4 o;
#pragma unroll
                    for (int e = 0; e < 4; ++e) o[e] = silu(s[e] * (1.f / WSCALE));
                    st16(__builtin_bit_cast(u32x4, o), ro, ((tl * 16 + r * 4 + (lane >> 4)) * NSH + (lane & 15) * 4) * 4);
                }
            }
            have = 0;
#pragma unroll
            for (int i = 0; i < 8; ++i) if (full(a[i])) have |= 1u << i;
        }
    }
    if (failed && lane == 0) __hip_atomic_store((GAS unsigned*)(myws + TMO_OFF), call, __ATOMIC_RELAXED, __HIP_MEMORY_SCOPE_AGENT);
    if (q >= 4) {
        wait_vm<0>();
        if (lane == 0 && lds_add(cnt + 7) == 7)
            __hip_atomic_store((GAS unsigned*)p.flagq[q - 4] + ((task >> 8) & 31), call, __ATOMIC_RELAXED, __HIP_MEMORY_SCOPE_SYSTEM);
    }
}

static bool build_tasks(int rank, u64* task) {
    struct RT { double t; int id; bool coop; };
    std::vector<RT> rts;
    for (int s = 0; s < NR; ++s) {
        const bool same = ((s ^ rank) & 1) == 0;
        const int n0 = same ? NPHI : NPSI, first = same ? 0 : NPHI;
        for (int i = 0; i < n0 + NRHO; ++i) {
            const int rt = i < n0 ? first + i : NPHI + NPSI + (i - n0);
            const double t = s == rank ? 0.05 * (i + 1) : same ? (i + 1) : 1.3 * (4 + i + 1);
            rts.push_back({t + 0.001 * ((s - rank) & 7), s * 16 + rt, i < n0});
        }
    }
    std::stable_sort(rts.begin(), rts.end(), [](const RT& a, const RT& b) { return a.t < b.t; });
    std::vector<RT> own[4], coop;
    { int n = 0; for (auto& r : rts) { own[n++ % 4].push_back(r); if (r.coop) coop.push_back(r); } }
    for (int X = 0; X < 8; ++X) {
        std::vector<int> lists[CPX];
        int qq[CPX];
        auto deal = [&](const std::vector<RT>& v, int s0, int nw, int rot) {
            const int n = (int)v.size(), extra = n % nw;
            int k = 0;
            for (; k < extra; ++k) lists[s0 + (k + rot) % nw].push_back(v[k].id);
            for (int i = 0; k < n; ++k, ++i) lists[s0 + (nw - 1 - i % nw + rot + extra) % nw].push_back(v[k].id);
        };
        if (X & 1) {
            deal(coop, 0, CPX, 0);
            for (int i = 0; i < CPX; ++i) qq[i] = 4 + (((rank + X) & 7) >> 1);
        } else {
            for (int q = 0; q < 4; ++q) {
                deal(own[X >> 1], q * (CPX / 4), CPX / 4, q);
                for (int i = 0; i < CPX / 4; ++i) qq[q * (CPX / 4) + i] = q;
            }
        }
        for (int i = 0; i < CPX; ++i) {
            if (lists[i].empty() || lists[i].size() > (size_t)MAXT) return false;
            u64 e = (u64)qq[i] | ((u64)lists[i].size() << 4) | ((u64)i << 8);
            for (size_t t = 0; t < lists[i].size(); ++t) e |= (u64)lists[i][t] << (16 + 7 * t);
            task[X + 8 * i] = e;
        }
    }
    return true;
}

extern "C" void kernel_launch(int rank, int nranks, ncclComm_t comm, hipStream_t stream,
                              void* const* d_in, const int* in_sizes, int n_in,
                              void* d_out, int out_size, const int* in_dtypes, int out_dtype,
                              void* const* d_out_all, void* const* d_ws_all, size_t ws_bytes) {
    if (nranks != NR || (unsigned)rank >= (unsigned)NR || ws_bytes < WS_NEED || n_in != 2 ||
        in_sizes[0] != MSH * KD || in_sizes[1] != KD * NSH || out_size != NR * MSH * NSH) {
        fprintf(stderr, "ag_gemm_silu: unexpected problem shape\n");
        return;
    }
    HIP_CHECK(hipFuncSetAttribute((const void*)ag_gemm_silu, hipFuncAttributeMaxDynamicSharedMemorySize, LDS_BYTES));
    Params p;
    p.x = (const float*)d_in[0]; p.w = (const float*)d_in[1];
    for (int i = 0; i < NR; ++i) p.peer[i] = (char*)d_ws_all[(rank + i) % NR];
    for (int q = 0; q < 4; ++q) {
        const int j = q * 2 + ((rank & 1) ^ 1);
        p.outq[q] = (float*)d_out + q * 64;
        p.outq[4 + q] = (float*)d_out_all[j] + (rank >> 1) * 64;
        p.flagq[q] = (unsigned*)((char*)d_ws_all[j] + FLAG_OFF) + (rank >> 1) * CPX;
    }
    p.rank = rank; p.pad = 0;
    if (!build_tasks(rank, p.task)) { fprintf(stderr, "ag_gemm_silu: bad schedule\n"); return; }
    hipLaunchKernelGGL(ag_gemm_silu, dim3(NWG), dim3(NTHREADS), LDS_BYTES, stream, p);
    HIP_CHECK(hipGetLastError());
}
```
